```python
import numpy as np

kernel_name = "gpu_gemm_rs_m4096_k4096_n16384_f32_qe4m3"


ABSMAX_RTOL = 0.09

COMPUTE_DTYPE = "float32"
EPILOGUE = "quant_e4m3"
IN_QUANT = None
OUT_QUANT = "e4m3"
IN_DTYPES = {"x": "float32", "w_mat": "float32"}
WORLD_SIZE = 8

def setup_inputs():

    x = np.random.randn(4096, 4096).astype(np.float32)
    w_mat = (np.random.randn(4096, 16384) / np.sqrt(4096)).astype(np.float32)
    return {"x": x, "w_mat": w_mat}

def _snap_fp8(x, mant_bits, max_abs, min_normal):
    a = np.abs(x.astype(np.float64))
    _m, _e = np.frexp(a)
    q = np.where(a >= min_normal, np.ldexp(1.0, _e - 1 - mant_bits),
                 min_normal * 2.0 ** -mant_bits)
    snapped = np.minimum(np.round(a / q) * q, max_abs)
    return (np.sign(x) * snapped).astype(np.float32)
def _quant_dequant(y, amax):
    scale = amax / 448.0
    q = _snap_fp8(y / scale, 3, 448.0, 2.0 ** -6)
    return (q * scale).astype(np.float32)
def _epilogue(y):
    return _quant_dequant(y, np.abs(y).max())

def reference(x, w_mat):
    y = (x @ w_mat).astype(np.float32)
    return _epilogue(y)
```

```cpp
#include <hip/hip_runtime.h>
#include <rccl/rccl.h>
#include <cstdio>
#include <cstdint>

typedef __attribute__((ext_vector_type(8))) short bf16x8;
typedef __attribute__((ext_vector_type(4))) float f32x4;
typedef __attribute__((ext_vector_type(4))) unsigned int u32x4;

namespace {
constexpr int R = 8, M = 4096, N = 16384, KS = 512, NS = 2048, MS = 512, KC = 64, J = 8;
constexpr int P = 256;
constexpr int G = 256;
constexpr int NT = 512;
constexpr int BM = 256, BN = 128;
constexpr size_t XW_OFF = 0, WW_OFF = 32ull << 20, CT_OFF = 48ull << 20;
constexpr int L_CALL = 0, L_TMO = 1, L_AMAXL = 2, L_AMAXT = 3, L_DONE = 4, L_PUSH = 8, L_AMAXS = 16, L_INF = 32;
constexpr unsigned long long DEADLINE = 100000000ull;
constexpr int LDS_BYTES = 48 * 1024;

__device__ __forceinline__ unsigned* ctl(void* ws, int line) {
  return reinterpret_cast<unsigned*>(static_cast<char*>(ws) + CT_OFF + (size_t)line * 128);
}
__device__ __forceinline__ unsigned pk_bf16(float a, float b) {
  unsigned ua = __float_as_uint(a), ub = __float_as_uint(b);
  ua = (ua + 0x7FFFu + ((ua >> 16) & 1u)) >> 16;
  ub = (ub + 0x7FFFu + ((ub >> 16) & 1u)) & 0xFFFF0000u;
  return ua | ub;
}
#define VMWAIT asm volatile("s_waitcnt vmcnt(0)" ::: "memory")

__device__ __forceinline__ bool wait_ge(const unsigned* flag, unsigned target, unsigned* tmo, unsigned call,
                                        unsigned long long t0) {
  while ((int)(__hip_atomic_load(flag, __ATOMIC_RELAXED, __HIP_MEMORY_SCOPE_SYSTEM) - target) < 0) {
    if (__hip_atomic_load(tmo, __ATOMIC_RELAXED, __HIP_MEMORY_SCOPE_AGENT) == call ||
        __builtin_amdgcn_s_memrealtime() - t0 > DEADLINE) {
      __hip_atomic_store(tmo, call, __ATOMIC_RELAXED, __HIP_MEMORY_SCOPE_AGENT);
      return false;
    }
    __builtin_amdgcn_s_sleep(1);
  }
  return true;
}
}

__global__ void __launch_bounds__(NT, 2)
fused_kernel(const float* __restrict__ xs, const float* __restrict__ wsrc, void* const* __restrict__ out_all,
             void* const* __restrict__ ws_all, int rank) {
  extern __shared__ __attribute__((aligned(16))) char lds[];
  __shared__ unsigned s_call;
  __shared__ float s_red[8];
  const int tid = threadIdx.x;
  char* myws = static_cast<char*>(ws_all[rank]);
  if (tid == 0) {
    unsigned t = __hip_atomic_fetch_add(ctl(myws, L_CALL), 1u, __ATOMIC_RELAXED, __HIP_MEMORY_SCOPE_AGENT);
    s_call = t / (unsigned)(P + G) + 1u;
  }
  __syncthreads();
  const unsigned call = s_call;
  unsigned* tmo = ctl(myws, L_TMO);

  if (blockIdx.x < P) {
    const int b = blockIdx.x;
    for (int j = 0; j < J; ++j) {
      const int kc = rank * J + j;
      {
        const int pc = tid & 127, dq = tid >> 7;
        const int row = b * 16 + (pc >> 3), p = pc & 7;
        const float4* src = reinterpret_cast<const float4*>(xs + (size_t)row * KS + j * KC + p * 8);
        float4 v0 = src[0], v1 = src[1];
        u32x4 o = {pk_bf16(v0.x, v0.y), pk_bf16(v0.z, v0.w), pk_bf16(v1.x, v1.y), pk_bf16(v1.z, v1.w)};
        const size_t off = XW_OFF + ((size_t)kc * M + row) * 128 + ((p ^ (row & 7)) * 16);
#pragma unroll
        for (int d = 0; d < 2; ++d) {
          int dest = (rank + 1 + dq * 2 + d) & 7;
          *reinterpret_cast<u32x4*>(static_cast<char*>(ws_all[dest]) + off) = o;
        }
      }
      {
        const int dest = b >> 5;
        const int nl = tid >> 3, p = tid & 7;
        const int n = b * 64 + nl;
        const float* src = wsrc + (size_t)(j * KC + p * 8) * N + n;
        float v[8];
#pragma unroll
        for (int e = 0; e < 8; ++e) v[e] = src[(size_t)e * N];
        u32x4 o = {pk_bf16(v[0], v[1]), pk_bf16(v[2], v[3]), pk_bf16(v[4], v[5]), pk_bf16(v[6], v[7])};
        const int np = n & (NS - 1);
        const size_t off = WW_OFF + ((size_t)kc * NS + np) * 128 + ((p ^ (np & 7)) * 16);
        *reinterpret_cast<u32x4*>(static_cast<char*>(ws_all[dest]) + off) = o;
      }
      VMWAIT;
      __syncthreads();
      if (tid == 0) {
        __builtin_amdgcn_fence(__ATOMIC_RELEASE, "");
        VMWAIT;
        unsigned t = __hip_atomic_fetch_add(ctl(myws, L_PUSH + j), 1u, __ATOMIC_RELEASE, __HIP_MEMORY_SCOPE_AGENT);
        if (t % (unsigned)P == (unsigned)(P - 1)) {
          __builtin_amdgcn_fence(__ATOMIC_ACQUIRE, "agent");
          __builtin_amdgcn_fence(__ATOMIC_RELEASE, "");
          VMWAIT;
          for (int d = 0; d < R; ++d)
            __hip_atomic_store(ctl(ws_all[d], L_INF + rank * J + j), call, __ATOMIC_RELAXED,
                               __HIP_MEMORY_SCOPE_SYSTEM);
        }
      }
    }
    return;
  }

  const int g = blockIdx.x - P;
  const int mt = g >> 4, nt = g & 15;
  const int wid = tid >> 6, lane = tid & 63, wm = wid >> 1, wn = wid & 1, fr = lane & 15, fq = lane >> 4;
  f32x4 acc[4][4];
#pragma unroll
  for (int a = 0; a < 4; ++a)
#pragma unroll
    for (int c = 0; c < 4; ++c) acc[a][c] = f32x4{0.f, 0.f, 0.f, 0.f};
  char* ldsA = lds;
  char* ldsB = lds + BM * 128;
  const unsigned long long t0 = __builtin_amdgcn_s_memrealtime();

  for (int j = 0; j < J; ++j) {
    if (tid < 64) {
      if (tid < R) wait_ge(ctl(myws, L_INF + tid * J + j), call, tmo, call, t0);
      __builtin_amdgcn_fence(__ATOMIC_ACQUIRE, "");
    }
    __syncthreads();
    u32x4 ra[4], rb[2];
    auto gload = [&](int src) {
      const int kc = src * J + j;
      const char* pa = myws + XW_OFF + ((size_t)kc * M + mt * BM) * 128;
      const char* pb = myws + WW_OFF + ((size_t)kc * NS + nt * BN) * 128;
#pragma unroll
      for (int i = 0; i < 4; ++i) ra[i] = *reinterpret_cast<const u32x4*>(pa + (i * NT + tid) * 16);
#pragma unroll
      for (int i = 0; i < 2; ++i) rb[i] = *reinterpret_cast<const u32x4*>(pb + (i * NT + tid) * 16);
    };
    gload(0);
    for (int s = 0; s < R; ++s) {
      __syncthreads();
#pragma unroll
      for (int i = 0; i < 4; ++i) *reinterpret_cast<u32x4*>(ldsA + (i * NT + tid) * 16) = ra[i];
#pragma unroll
      for (int i = 0; i < 2; ++i) *reinterpret_cast<u32x4*>(ldsB + (i * NT + tid) * 16) = rb[i];
      __syncthreads();
      if (s + 1 < R) gload(s + 1);
#pragma unroll
      for (int ks = 0; ks < 2; ++ks) {
        bf16x8 af[4], bfr[4];
        const int pc = ks * 4 + fq;
#pragma unroll
        for (int a = 0; a < 4; ++a) {
          const int row = wm * 64 + a * 16 + fr;
          af[a] = *reinterpret_cast<const bf16x8*>(ldsA + row * 128 + ((pc ^ (row & 7)) * 16));
        }
#pragma unroll
        for (int c = 0; c < 4; ++c) {
          const int row = wn * 64 + c * 16 + fr;
          bfr[c] = *reinterpret_cast<const bf16x8*>(ldsB + row * 128 + ((pc ^ (row & 7)) * 16));
        }
#pragma unroll
        for (int a = 0; a < 4; ++a)
#pragma unroll
          for (int c = 0; c < 4; ++c)
            acc[a][c] = __builtin_amdgcn_mfma_f32_16x16x32_bf16(af[a], bfr[c], acc[a][c], 0, 0, 0);
      }
    }
  }

  float mx = 0.f;
#pragma unroll
  for (int a = 0; a < 4; ++a)
#pragma unroll
    for (int c = 0; c < 4; ++c)
#pragma unroll
      for (int e = 0; e < 4; ++e) mx = fmaxf(mx, fabsf(acc[a][c][e]));
#pragma unroll
  for (int o = 32; o >= 1; o >>= 1) mx = fmaxf(mx, __shfl_xor(mx, o));
  if (lane == 0) s_red[wid] = mx;
  __syncthreads();
  if (tid == 0) {
    float m = s_red[0];
    for (int i = 1; i < 8; ++i) m = fmaxf(m, s_red[i]);
    unsigned long long key = ((unsigned long long)call << 32) | (unsigned long long)__float_as_uint(m);
    unsigned long long* al = reinterpret_cast<unsigned long long*>(ctl(myws, L_AMAXL));
    __hip_atomic_fetch_max(al, key, __ATOMIC_RELAXED, __HIP_MEMORY_SCOPE_AGENT);
    unsigned t = __hip_atomic_fetch_add(ctl(myws, L_AMAXT), 1u, __ATOMIC_ACQ_REL, __HIP_MEMORY_SCOPE_AGENT);
    if (t % (unsigned)G == (unsigned)(G - 1)) {
      unsigned long long v = __hip_atomic_load(al, __ATOMIC_RELAXED, __HIP_MEMORY_SCOPE_AGENT);
      for (int d = 0; d < R; ++d)
        __hip_atomic_store(reinterpret_cast<unsigned long long*>(ctl(ws_all[d], L_AMAXS + rank)), v,
                           __ATOMIC_RELAXED, __HIP_MEMORY_SCOPE_SYSTEM);
    }
  }
  if (tid < 64) {
    float m = 0.f;
    if (tid < R) {
      const unsigned* hi = ctl(myws, L_AMAXS + tid) + 1;
      wait_ge(hi, call, tmo, call, t0);
      unsigned long long v = __hip_atomic_load(reinterpret_cast<unsigned long long*>(ctl(myws, L_AMAXS + tid)),
                                               __ATOMIC_RELAXED, __HIP_MEMORY_SCOPE_SYSTEM);
      m = __uint_as_float((unsigned)v);
    }
#pragma unroll
    for (int o = 4; o >= 1; o >>= 1) m = fmaxf(m, __shfl_xor(m, o));
    if (tid == 0) s_red[0] = m;
  }
  __syncthreads();
  const float scale = s_red[0] / 448.0f;
  const bool bad = __hip_atomic_load(tmo, __ATOMIC_RELAXED, __HIP_MEMORY_SCOPE_AGENT) == call;
#pragma unroll
  for (int a = 0; a < 4; ++a)
#pragma unroll
    for (int c = 0; c < 4; ++c) {
      const int col = wn * 64 + c * 16 + fr;
      const int row0 = wm * 64 + a * 16 + fq * 4;
      int q01 = __builtin_amdgcn_cvt_pk_fp8_f32(acc[a][c][0] / scale, acc[a][c][1] / scale, 0, false);
      int q23 = __builtin_amdgcn_cvt_pk_fp8_f32(acc[a][c][2] / scale, acc[a][c][3] / scale, 0, false);
      lds[(row0 + 0) * 128 + col] = (char)(q01 & 0xFF);
      lds[(row0 + 1) * 128 + col] = (char)((q01 >> 8) & 0xFF);
      lds[(row0 + 2) * 128 + col] = (char)(q23 & 0xFF);
      lds[(row0 + 3) * 128 + col] = (char)((q23 >> 8) & 0xFF);
    }
  __syncthreads();
  const int owner = mt >> 1;
  char* od = static_cast<char*>(out_all[owner]);
#pragma unroll
  for (int i = 0; i < 4; ++i) {
    const int idx = i * NT + tid, row = idx >> 3, p = idx & 7;
    u32x4 v = *reinterpret_cast<const u32x4*>(lds + row * 128 + p * 16);
    if (bad) v = u32x4{0x7F7F7F7Fu, 0x7F7F7F7Fu, 0x7F7F7F7Fu, 0x7F7F7F7Fu};
    *reinterpret_cast<u32x4*>(od + (size_t)((mt & 1) * BM + row) * N + rank * NS + nt * BN + p * 16) = v;
  }
  VMWAIT;
  __syncthreads();
  if (tid == 0) {
    __builtin_amdgcn_fence(__ATOMIC_RELEASE, "");
    VMWAIT;
    __hip_atomic_fetch_add(ctl(ws_all[owner], L_DONE), 1u, __ATOMIC_RELAXED, __HIP_MEMORY_SCOPE_SYSTEM);
    if (g == 0) {
      if (!wait_ge(ctl(myws, L_DONE), call * 256u, tmo, call, t0))
        static_cast<unsigned char*>(out_all[rank])[0] = 0x7F;
      __builtin_amdgcn_fence(__ATOMIC_ACQUIRE, "");
    }
  }
}

extern "C" void kernel_launch(int rank, int nranks, ncclComm_t comm, hipStream_t stream, void* const* d_in,
                              const int* in_sizes, int n_in, void* d_out, int out_size, const int* in_dtypes,
                              int out_dtype, void* const* d_out_all, void* const* d_ws_all, size_t ws_bytes) {
  fused_kernel<<<P + G, NT, LDS_BYTES, stream>>>(static_cast<const float*>(d_in[0]),
                                                 static_cast<const float*>(d_in[1]), d_out_all, d_ws_all, rank);
}
```

```cpp
#include <hip/hip_runtime.h>
#include <rccl/rccl.h>
#include <cstdio>
#include <cstdint>

typedef __attribute__((ext_vector_type(8))) short bf16x8;
typedef __attribute__((ext_vector_type(4))) float f32x4;
typedef __attribute__((ext_vector_type(4))) unsigned int u32x4;

namespace {
constexpr int R = 8, M = 4096, N = 16384, KS = 512, NS = 2048, MS = 512, KC = 64, J = 8;
#define PJV 64
constexpr int PJ = PJV;
constexpr int P = PJ * 8;
constexpr int G = 256;
constexpr int NT = 512;
constexpr int BM = 256, BN = 128;
constexpr size_t XW_OFF = 0, WW_OFF = 32ull << 20, CT_OFF = 48ull << 20;
constexpr int L_CALL = 0, L_TMO = 1, L_AMAXL = 2, L_AMAXT = 3, L_DONE = 4, L_PUSH = 8, L_AMAXS = 16, L_INF = 32;
constexpr unsigned long long DEADLINE = 100000000ull;
constexpr int LDS_BYTES = 48 * 1024;

__device__ __forceinline__ unsigned* ctl(void* ws, int line) {
  return reinterpret_cast<unsigned*>(static_cast<char*>(ws) + CT_OFF + (size_t)line * 128);
}
__device__ __forceinline__ unsigned pk_bf16(float a, float b) {
  unsigned ua = __float_as_uint(a), ub = __float_as_uint(b);
  ua = (ua + 0x7FFFu + ((ua >> 16) & 1u)) >> 16;
  ub = (ub + 0x7FFFu + ((ub >> 16) & 1u)) & 0xFFFF0000u;
  return ua | ub;
}
#define VMWAIT asm volatile("s_waitcnt vmcnt(0)" ::: "memory")
#define STAMP(i) do {} while (0)

__device__ __forceinline__ bool wait_ge(const unsigned* flag, unsigned target, unsigned* tmo, unsigned call,
                                        unsigned long long t0) {
  while ((int)(__hip_atomic_load(flag, __ATOMIC_RELAXED, __HIP_MEMORY_SCOPE_SYSTEM) - target) < 0) {
    if (__hip_atomic_load(tmo, __ATOMIC_RELAXED, __HIP_MEMORY_SCOPE_AGENT) == call ||
        __builtin_amdgcn_s_memrealtime() - t0 > DEADLINE) {
      __hip_atomic_store(tmo, call, __ATOMIC_RELAXED, __HIP_MEMORY_SCOPE_AGENT);
      return false;
    }
    __builtin_amdgcn_s_sleep(1);
  }
  return true;
}
}

__global__ void __launch_bounds__(NT, 4)
fused_kernel(const float* __restrict__ xs, const float* __restrict__ wsrc, void* const* __restrict__ out_all,
             void* const* __restrict__ ws_all, int rank) {
  extern __shared__ __attribute__((aligned(16))) char lds[];
  __shared__ unsigned s_call;
  __shared__ float s_red[8];
  const int tid = threadIdx.x;
  char* myws = static_cast<char*>(ws_all[rank]);
  if (tid == 0) {
    unsigned t = __hip_atomic_fetch_add(ctl(myws, L_CALL), 1u, __ATOMIC_RELAXED, __HIP_MEMORY_SCOPE_AGENT);
    s_call = t / (unsigned)(P + G) + 1u;
  }
  __syncthreads();
  const unsigned call = s_call;
  unsigned* tmo = ctl(myws, L_TMO);
  STAMP(0);

  if (blockIdx.x >= G) {
    const int q = blockIdx.x - G, j = q / PJ, sl = q % PJ;
    const int kc = rank * J + j;
    {
      constexpr int XR = M / PJ, XP = XR * 8, DS = NT / XP, DN = R / DS;
      const int pc = tid % XP, dq = tid / XP;
      const int row = sl * XR + (pc >> 3), p = pc & 7;
      const float4* src = reinterpret_cast<const float4*>(xs + (size_t)row * KS + j * KC + p * 8);
      float4 v0 = src[0], v1 = src[1];
      u32x4 o = {pk_bf16(v0.x, v0.y), pk_bf16(v0.z, v0.w), pk_bf16(v1.x, v1.y), pk_bf16(v1.z, v1.w)};
      const size_t off = XW_OFF + ((size_t)kc * M + row) * 128 + ((p ^ (row & 7)) * 16);
#pragma unroll
      for (int d = 0; d < DN; ++d)
        *reinterpret_cast<u32x4*>(static_cast<char*>(ws_all[(rank + 1 + dq * DN + d) & 7]) + off) = o;
    }
    {
      constexpr int WC = N / PJ, WI = WC / 64;
      char* dst = static_cast<char*>(ws_all[sl / (PJ / 8)]);
      const int p = tid & 7;
      float v[WI][8];
#pragma unroll
      for (int i = 0; i < WI; ++i) {
        const int n = sl * WC + i * 64 + (tid >> 3);
        const float* src = wsrc + (size_t)(j * KC + p * 8) * N + n;
#pragma unroll
        for (int e = 0; e < 8; ++e) v[i][e] = src[(size_t)e * N];
      }
#pragma unroll
      for (int i = 0; i < WI; ++i) {
        const int np = (sl * WC + i * 64 + (tid >> 3)) & (NS - 1);
        u32x4 o = {pk_bf16(v[i][0], v[i][1]), pk_bf16(v[i][2], v[i][3]), pk_bf16(v[i][4], v[i][5]),
                   pk_bf16(v[i][6], v[i][7])};
        *reinterpret_cast<u32x4*>(dst + WW_OFF + ((size_t)kc * NS + np) * 128 + ((p ^ (np & 7)) * 16)) = o;
      }
    }
    VMWAIT;
    __syncthreads();
    if (tid == 0) {
      __builtin_amdgcn_fence(__ATOMIC_RELEASE, "");
      VMWAIT;
      unsigned t = __hip_atomic_fetch_add(ctl(myws, L_PUSH + j), 1u, __ATOMIC_RELEASE, __HIP_MEMORY_SCOPE_AGENT);
      if (t % (unsigned)PJ == (unsigned)(PJ - 1)) {
        __builtin_amdgcn_fence(__ATOMIC_ACQUIRE, "agent");
        __builtin_amdgcn_fence(__ATOMIC_RELEASE, "");
        VMWAIT;
        for (int d = 0; d < R; ++d)
          __hip_atomic_store(ctl(ws_all[d], L_INF + rank * J + j), call, __ATOMIC_RELAXED,
                             __HIP_MEMORY_SCOPE_SYSTEM);
      }
    }
    STAMP(1);
    return;
  }

  const int g = blockIdx.x;
  const int mt = g >> 4, nt = g & 15;
  const int wid = tid >> 6, lane = tid & 63, wm = wid >> 1, wn = wid & 1, fr = lane & 15, fq = lane >> 4;
  f32x4 acc[4][4];
#pragma unroll
  for (int a = 0; a < 4; ++a)
#pragma unroll
    for (int c = 0; c < 4; ++c) acc[a][c] = f32x4{0.f, 0.f, 0.f, 0.f};
  char* ldsA = lds;
  char* ldsB = lds + BM * 128;
  const unsigned long long t0 = __builtin_amdgcn_s_memrealtime();

  for (int j = 0; j < J; ++j) {
    if (tid < 64) {
      if (tid < R) wait_ge(ctl(myws, L_INF + tid * J + j), call, tmo, call, t0);
      __builtin_amdgcn_fence(__ATOMIC_ACQUIRE, "");
    }
    __syncthreads();
    STAMP(1 + j);
    u32x4 ra[4], rb[2];
    auto gload = [&](int src) {
      const int kc = src * J + j;
      const char* pa = myws + XW_OFF + ((size_t)kc * M + mt * BM) * 128;
      const char* pb = myws + WW_OFF + ((size_t)kc * NS + nt * BN) * 128;
#pragma unroll
      for (int i = 0; i < 4; ++i) ra[i] = *reinterpret_cast<const u32x4*>(pa + (i * NT + tid) * 16);
#pragma unroll
      for (int i = 0; i < 2; ++i) rb[i] = *reinterpret_cast<const u32x4*>(pb + (i * NT + tid) * 16);
    };
    gload(0);
    for (int s = 0; s < R; ++s) {
      __syncthreads();
#pragma unroll
      for (int i = 0; i < 4; ++i) *reinterpret_cast<u32x4*>(ldsA + (i * NT + tid) * 16) = ra[i];
#pragma unroll
      for (int i = 0; i < 2; ++i) *reinterpret_cast<u32x4*>(ldsB + (i * NT + tid) * 16) = rb[i];
      __syncthreads();
      if (s + 1 < R) gload(s + 1);
#pragma unroll
      for (int ks = 0; ks < 2; ++ks) {
        bf16x8 af[4], bfr[4];
        const int pc = ks * 4 + fq;
#pragma unroll
        for (int a = 0; a < 4; ++a) {
          const int row = wm * 64 + a * 16 + fr;
          af[a] = *reinterpret_cast<const bf16x8*>(ldsA + row * 128 + ((pc ^ (row & 7)) * 16));
        }
#pragma unroll
        for (int c = 0; c < 4; ++c) {
          const int row = wn * 64 + c * 16 + fr;
          bfr[c] = *reinterpret_cast<const bf16x8*>(ldsB + row * 128 + ((pc ^ (row & 7)) * 16));
        }
#pragma unroll
        for (int a = 0; a < 4; ++a)
#pragma unroll
          for (int c = 0; c < 4; ++c)
            acc[a][c] = __builtin_amdgcn_mfma_f32_16x16x32_bf16(af[a], bfr[c], acc[a][c], 0, 0, 0);
      }
    }
  }

  STAMP(9);
  float mx = 0.f;
#pragma unroll
  for (int a = 0; a < 4; ++a)
#pragma unroll
    for (int c = 0; c < 4; ++c)
#pragma unroll
      for (int e = 0; e < 4; ++e) mx = fmaxf(mx, fabsf(acc[a][c][e]));
#pragma unroll
  for (int o = 32; o >= 1; o >>= 1) mx = fmaxf(mx, __shfl_xor(mx, o));
  if (lane == 0) s_red[wid] = mx;
  __syncthreads();
  if (tid == 0) {
    float m = s_red[0];
    for (int i = 1; i < 8; ++i) m = fmaxf(m, s_red[i]);
    unsigned long long key = ((unsigned long long)call << 32) | (unsigned long long)__float_as_uint(m);
    unsigned long long* al = reinterpret_cast<unsigned long long*>(ctl(myws, L_AMAXL));
    __hip_atomic_fetch_max(al, key, __ATOMIC_RELAXED, __HIP_MEMORY_SCOPE_AGENT);
    unsigned t = __hip_atomic_fetch_add(ctl(myws, L_AMAXT), 1u, __ATOMIC_ACQ_REL, __HIP_MEMORY_SCOPE_AGENT);
    if (t % (unsigned)G == (unsigned)(G - 1)) {
      unsigned long long v = __hip_atomic_load(al, __ATOMIC_RELAXED, __HIP_MEMORY_SCOPE_AGENT);
      for (int d = 0; d < R; ++d)
        __hip_atomic_store(reinterpret_cast<unsigned long long*>(ctl(ws_all[d], L_AMAXS + rank)), v,
                           __ATOMIC_RELAXED, __HIP_MEMORY_SCOPE_SYSTEM);
    }
  }
  if (tid < 64) {
    float m = 0.f;
    if (tid < R) {
      const unsigned* hi = ctl(myws, L_AMAXS + tid) + 1;
      wait_ge(hi, call, tmo, call, t0);
      unsigned long long v = __hip_atomic_load(reinterpret_cast<unsigned long long*>(ctl(myws, L_AMAXS + tid)),
                                               __ATOMIC_RELAXED, __HIP_MEMORY_SCOPE_SYSTEM);
      m = __uint_as_float((unsigned)v);
    }
#pragma unroll
    for (int o = 4; o >= 1; o >>= 1) m = fmaxf(m, __shfl_xor(m, o));
    if (tid == 0) s_red[0] = m;
  }
  __syncthreads();
  STAMP(10);
  const float scale = s_red[0] / 448.0f;
  const bool bad = __hip_atomic_load(tmo, __ATOMIC_RELAXED, __HIP_MEMORY_SCOPE_AGENT) == call;
#pragma unroll
  for (int a = 0; a < 4; ++a)
#pragma unroll
    for (int c = 0; c < 4; ++c) {
      const int col = wn * 64 + c * 16 + fr;
      const int row0 = wm * 64 + a * 16 + fq * 4;
      int q01 = __builtin_amdgcn_cvt_pk_fp8_f32(acc[a][c][0] / scale, acc[a][c][1] / scale, 0, false);
      int q23 = __builtin_amdgcn_cvt_pk_fp8_f32(acc[a][c][2] / scale, acc[a][c][3] / scale, 0, false);
      lds[(row0 + 0) * 128 + col] = (char)(q01 & 0xFF);
      lds[(row0 + 1) * 128 + col] = (char)((q01 >> 8) & 0xFF);
      lds[(row0 + 2) * 128 + col] = (char)(q23 & 0xFF);
      lds[(row0 + 3) * 128 + col] = (char)((q23 >> 8) & 0xFF);
    }
  __syncthreads();
  const int owner = mt >> 1;
  char* od = static_cast<char*>(out_all[owner]);
#pragma unroll
  for (int i = 0; i < 4; ++i) {
    const int idx = i * NT + tid, row = idx >> 3, p = idx & 7;
    u32x4 v = *reinterpret_cast<const u32x4*>(lds + row * 128 + p * 16);
    if (bad) v = u32x4{0x7F7F7F7Fu, 0x7F7F7F7Fu, 0x7F7F7F7Fu, 0x7F7F7F7Fu};
    *reinterpret_cast<u32x4*>(od + (size_t)((mt & 1) * BM + row) * N + rank * NS + nt * BN + p * 16) = v;
  }
  VMWAIT;
  __syncthreads();
  STAMP(11);
  if (tid == 0) {
    __builtin_amdgcn_fence(__ATOMIC_RELEASE, "");
    VMWAIT;
    __hip_atomic_fetch_add(ctl(ws_all[owner], L_DONE), 1u, __ATOMIC_RELAXED, __HIP_MEMORY_SCOPE_SYSTEM);
    if (g == 0) {
      if (!wait_ge(ctl(myws, L_DONE), call * 256u, tmo, call, t0))
        static_cast<unsigned char*>(out_all[rank])[0] = 0x7F;
      __builtin_amdgcn_fence(__ATOMIC_ACQUIRE, "");
    }
  }
  STAMP(12);
}

extern "C" void kernel_launch(int rank, int nranks, ncclComm_t comm, hipStream_t stream, void* const* d_in,
                              const int* in_sizes, int n_in, void* d_out, int out_size, const int* in_dtypes,
                              int out_dtype, void* const* d_out_all, void* const* d_ws_all, size_t ws_bytes) {
  fused_kernel<<<P + G, NT, LDS_BYTES, stream>>>(static_cast<const float*>(d_in[0]),
                                                 static_cast<const float*>(d_in[1]), d_out_all, d_ws_all, rank);
}
```
